```python
import numpy as np

kernel_name = "gpu_ag_gemm_m4096_k4096_n16384_bf16_silu"


ABSMAX_RTOL = 0.008

COMPUTE_DTYPE = "bfloat16"
EPILOGUE = "silu"
IN_QUANT = None
OUT_QUANT = None
IN_DTYPES = {"x": "bfloat16", "w_mat": "bfloat16"}
WORLD_SIZE = 8

def _to_bf16(x):
    u = np.ascontiguousarray(x, dtype=np.float32).view(np.uint32)
    return ((u + 0x7FFF + ((u >> 16) & 1)) & 0xFFFF0000).view(np.float32)

def setup_inputs():

    x = _to_bf16(np.random.randn(4096, 4096).astype(np.float32))
    w_mat = _to_bf16((np.random.randn(4096, 16384) / np.sqrt(4096)).astype(np.float32))
    return {"x": x, "w_mat": w_mat}

def _epilogue(y):
    return (y / (1.0 + np.exp(-np.clip(y, -60.0, 60.0)))).astype(np.float32)

def reference(x, w_mat):
    y = (x @ w_mat).astype(np.float32)
    return _epilogue(y)
```

```cpp
#include <hip/hip_runtime.h>
#include <rccl/rccl.h>
#include <cstdio>
#include <cstdint>

#define HIP_CHECK(expr) do { hipError_t _e = (expr); if (_e != hipSuccess) { \
    fprintf(stderr, "HIP error %s at %s:%d\n", hipGetErrorString(_e), __FILE__, __LINE__); \
    return; } } while (0)

typedef __attribute__((ext_vector_type(8))) short bf16x8;
typedef __attribute__((ext_vector_type(4))) short bf16x4;
typedef __attribute__((ext_vector_type(4))) float f32x4;
typedef __attribute__((ext_vector_type(2))) float f32x2;
typedef __attribute__((ext_vector_type(4))) unsigned u32x4;
typedef __attribute__((ext_vector_type(2))) unsigned u32x2;
typedef __attribute__((ext_vector_type(2))) __bf16 bf16x2_t;

namespace {

constexpr int NR = 8;
constexpr int KTOT = 4096, NLOC = 2048, MLOC = 512;
constexpr int BK = 64, NKS = KTOT / BK;
constexpr int NWG = 256, NCOMM = 32;
#define AG_NWPP 4
constexpr int NWPP = AG_NWPP;
static_assert(7 * NWPP <= NCOMM, "comm blocks");
constexpr int XROW_B = KTOT * 2;
constexpr int WROW_B = NLOC * 2;

constexpr int A_B = 32768, B_B = 16384, STAGE_B = A_B + B_B;
constexpr int FLG_LDS = 3 * STAGE_B;
constexpr int MISC_LDS = FLG_LDS + 2048;
constexpr int LDS_BYTES = MISC_LDS + 64;

constexpr size_t CTR_OFF = 0;
constexpr size_t TMO_OFF = 32768;
constexpr size_t FLAG_OFF = 65536;
constexpr size_t ABUF_OFF = 1u << 20;
constexpr size_t SRC_B = 4u << 20, PAR_B = 32u << 20, KS_B = 65536;
constexpr size_t WS_NEED = ABUF_OFF + 2 * PAR_B;

constexpr unsigned long long DEADLINE_TICKS = 50000000ull;

struct Params {
  const char* x;
  const char* w;
  char* out;
  char* ws[NR];
  int rank;
};

#define WAIT_V(n) asm volatile("s_waitcnt vmcnt(" #n ")" ::: "memory")
#define WAIT_VN(n) asm volatile("s_waitcnt vmcnt(%0)" :: "n"(n) : "memory")
#define BAR() __builtin_amdgcn_s_barrier()
#define SCHED() __builtin_amdgcn_sched_barrier(0)
#define DSR128(dst, addr, off) asm volatile("ds_read_b128 %0, %1 offset:" #off : "=v"(dst) : "v"(addr))
#define DSRTR(dst, addr, off) asm volatile("ds_read_b64_tr_b16 %0, %1 offset:" #off : "=v"(dst) : "v"(addr))

__device__ __forceinline__ float silu_f(float y) {
  float c = fminf(fmaxf(y, -60.f), 60.f);
  float e = __expf(-c);
  return y * __builtin_amdgcn_rcpf(1.f + e);
}

template <int NI>
__device__ __forceinline__ void run_unit(char* smem, const unsigned lds0, const int tid, const int lane, const int wid,
                                         const unsigned call, unsigned* tmo,
                                         const char* abase, const unsigned akstr, const unsigned* aoff,
                                         const char* bbase, const unsigned* boff, const unsigned* flg,
                                         char* otile) {
  constexpr int NPA = (NI == 4) ? 4 : 2;
  constexpr int NP = NPA + 2;
  constexpr int NH = NP / 2;
  constexpr int NRD = 4 + 2 * NI;
  const int wm = (NI == 4) ? (wid >> 1) : (wid >> 2);
  const int wn = (NI == 4) ? (wid & 1) : (wid & 3);

  unsigned a_rd[2], b_rd[NI];
  {
    const unsigned r = (unsigned)(lane & 15), g = (unsigned)(lane >> 4), x = (unsigned)((lane >> 1) & 7);
    const unsigned base = (unsigned)(wm * 64 + r) * 128u;
    a_rd[0] = base + ((g ^ x) * 16u);
    a_rd[1] = base + (((4u + g) ^ x) * 16u);
    const unsigned q = (unsigned)((lane & 15) >> 2), pp = (unsigned)(lane & 3);
    const unsigned k = 8u * g + q;
    const unsigned h = q | ((g & 1u) << 2);
#pragma unroll
    for (int ni = 0; ni < NI; ++ni) {
      const unsigned cc = (unsigned)(wn * (2 * NI) + ni * 2) + (pp >> 1);
      b_rd[ni] = (unsigned)A_B + k * 256u + ((cc ^ (h << 1)) * 16u) + (pp & 1u) * 8u;
    }
  }

  f32x4 acc[4][NI];
#pragma unroll
  for (int mi = 0; mi < 4; ++mi)
#pragma unroll
    for (int ni = 0; ni < NI; ++ni) acc[mi][ni] = f32x4{0.f, 0.f, 0.f, 0.f};

  int issued = 0;
  int ready = flg ? 0 : NKS;
  bool pend = false;
  int fill = -1;
  char* const myflg = smem + FLG_LDS + wid * 256 + lane * 4;
  const unsigned myflg_a = lds0 + (unsigned)(FLG_LDS + wid * 256 + lane * 4);

  auto gl = [&](int s, int i) {
    char* lb_ = smem + (s % 3) * STAGE_B + tid * 16;
    if (i < NPA) {
      const char* ab = abase + (size_t)s * akstr;
      __builtin_amdgcn_global_load_lds((const unsigned*)(ab + aoff[i]), (unsigned*)(lb_ + i * 8192), 16, 0, 0);
    } else {
      const char* bb = bbase + (size_t)s * (BK * WROW_B);
      __builtin_amdgcn_global_load_lds((const unsigned*)(bb + boff[i - NPA]), (unsigned*)(lb_ + A_B + (i - NPA) * 8192), 16, 0, 0);
    }
  };
  auto issue = [&](int s) {
#pragma unroll
    for (int i = 0; i < NP; ++i) gl(s, i);
  };
  auto poll_issue = [&]() {
    __builtin_amdgcn_global_load_lds(flg + lane, (unsigned*)myflg, 4, 0, 17);
  };
  auto poll_consume = [&]() {
    unsigned v;
    asm volatile("ds_read_b32 %0, %1\n\ts_waitcnt lgkmcnt(0)" : "=&v"(v) : "v"(myflg_a) : "memory");
    const unsigned long long m = __ballot(v >= call);
    ready = (m == ~0ull) ? NKS : (int)__builtin_ctzll(~m);
  };

  bf16x8 xa[4], ya[4];
  bf16x4 xl[NI], xh[NI], yl[NI], yh[NI];
#define READ_FRAGS(A_, L_, H_, st_, kk_, koff_) do {                                             \
    const unsigned sbo_ = lds0 + (unsigned)((st_) % 3) * (unsigned)STAGE_B;                         \
    const unsigned aa_ = sbo_ + a_rd[kk_];                                                          \
    DSR128(A_[0], aa_, 0); DSR128(A_[1], aa_, 2048); DSR128(A_[2], aa_, 4096); DSR128(A_[3], aa_, 6144); \
    _Pragma("unroll") for (int ni = 0; ni < NI; ++ni) {                                            \
      const unsigned bb_ = sbo_ + b_rd[ni];                                                         \
      DSRTR(L_[ni], bb_, koff_); DSRTR(H_[ni], bb_, koff_ + 1024); }                                \
  } while (0)
#define WAIT_FRAGS(n, A_, L_, H_) do {                                                            \
    if constexpr (NI == 4)                                                                          \
      asm volatile("s_waitcnt lgkmcnt(%12)" : "+v"(A_[0]), "+v"(A_[1]), "+v"(A_[2]), "+v"(A_[3]),  \
        "+v"(L_[0]), "+v"(L_[1]), "+v"(L_[2]), "+v"(L_[3]), "+v"(H_[0]), "+v"(H_[1]), "+v"(H_[2]), "+v"(H_[3]) : "n"(n)); \
    else                                                                                            \
      asm volatile("s_waitcnt lgkmcnt(%8)" : "+v"(A_[0]), "+v"(A_[1]), "+v"(A_[2]), "+v"(A_[3]),   \
        "+v"(L_[0]), "+v"(L_[1]), "+v"(H_[0]), "+v"(H_[1]) : "n"(n));                               \
  } while (0)
#define MMA_BLOCK(A_, L_, H_, i0_) do { __builtin_amdgcn_s_setprio(1);                            \
    _Pragma("unroll") for (int mi = 0; mi < 4; ++mi) {                                             \
      _Pragma("unroll") for (int ni = 0; ni < NI; ++ni) {                                          \
        const bf16x8 bb_ = __builtin_shufflevector(L_[ni], H_[ni], 0, 1, 2, 3, 4, 5, 6, 7);       \
        acc[mi][ni] = __builtin_amdgcn_mfma_f32_16x16x32_bf16(bb_, A_[mi], acc[mi][ni], 0, 0, 0); } \
      if (mi < NH) { SCHED(); if (fill >= 0) gl(fill, (i0_) + mi); SCHED(); } }                     \
    __builtin_amdgcn_s_setprio(0); } while (0)

  auto advance = [&](int t) {
    if (t + 1 < NKS) {
      if (issued <= t + 1) {
        WAIT_V(0);
        if (pend) { poll_consume(); pend = false; }
        if (ready <= issued) {
          const unsigned long long t0 = __builtin_amdgcn_s_memrealtime();
          for (;;) {
            poll_issue();
            WAIT_V(0);
            poll_consume();
            if (ready > issued) break;
            if (__hip_atomic_load(tmo, __ATOMIC_RELAXED, __HIP_MEMORY_SCOPE_AGENT) == call ||
                __builtin_amdgcn_s_memrealtime() - t0 > DEADLINE_TICKS) {
              __hip_atomic_store(tmo, call, __ATOMIC_RELAXED, __HIP_MEMORY_SCOPE_AGENT);
              ready = NKS;
              break;
            }
            __builtin_amdgcn_s_sleep(2);
          }
        }
        while (issued <= t + 2 && issued < ready) { issue(issued); ++issued; }
      }
      if (issued - 1 >= t + 2) { WAIT_VN(NP); } else { WAIT_V(0); }
      if (pend) { poll_consume(); pend = false; }
    }
    BAR();
    if (ready < NKS) { poll_issue(); pend = true; }
    if (t >= 0) {
      if (issued <= t + 3 && issued < ready) fill = issued;
    } else {
      while (issued <= t + 3 && issued < ready) { issue(issued); ++issued; }
    }
    if (t + 1 < NKS) READ_FRAGS(xa, xl, xh, t + 1, 0, 0);
  };

  while (issued < 2 && issued < ready) { issue(issued); ++issued; }
  advance(-1);
  for (int t = 0; t < NKS; ++t) {
    READ_FRAGS(ya, yl, yh, t, 1, 8192);
    WAIT_FRAGS(NRD, xa, xl, xh);
    MMA_BLOCK(xa, xl, xh, NH);
    if (fill >= 0) { ++issued; fill = -1; }
    while (issued <= t + 2 && issued < ready) { issue(issued); ++issued; }
    WAIT_FRAGS(0, ya, yl, yh);
    advance(t);
    MMA_BLOCK(ya, yl, yh, 0);
  }

  const bool bad = __hip_atomic_load(tmo, __ATOMIC_RELAXED, __HIP_MEMORY_SCOPE_AGENT) == call;
  char* obase = otile + ((size_t)(wm * 64 + (lane & 15)) * NLOC + (size_t)(wn * (16 * NI) + (lane >> 4) * 4)) * 2;
#pragma unroll
  for (int mi = 0; mi < 4; ++mi)
#pragma unroll
    for (int ni = 0; ni < NI; ++ni) {
      const f32x4 a = acc[mi][ni];
      const bf16x2_t lo = __builtin_convertvector(f32x2{silu_f(a[0]), silu_f(a[1])}, bf16x2_t);
      const bf16x2_t hi = __builtin_convertvector(f32x2{silu_f(a[2]), silu_f(a[3])}, bf16x2_t);
      u32x2 o;
      o[0] = __builtin_bit_cast(unsigned, lo);
      o[1] = __builtin_bit_cast(unsigned, hi);
      if (bad) { o[0] = 0x7FC07FC0u; o[1] = 0x7FC07FC0u; }
      *(u32x2*)(obase + (size_t)(mi * 16) * NLOC * 2 + ni * 32) = o;
    }
#undef READ_FRAGS
#undef WAIT_FRAGS
#undef MMA_BLOCK
}

}

__global__ __launch_bounds__(512, 2) void ag_gemm_silu_kernel(Params p) {
  extern __shared__ __attribute__((aligned(16))) char smem[];
  const int tid = threadIdx.x;
  const int bid = blockIdx.x;
  const int lane = tid & 63;
  const int wid = __builtin_amdgcn_readfirstlane(tid >> 6);
  char* const myws = p.ws[p.rank];

#define STAMP(i) do { } while (0)
  STAMP(0);
  if (tid == 0) {
    unsigned* c = (unsigned*)(myws + CTR_OFF + (size_t)bid * 128);
    unsigned v = __hip_atomic_load(c, __ATOMIC_RELAXED, __HIP_MEMORY_SCOPE_AGENT) + 1u;
    __hip_atomic_store(c, v, __ATOMIC_RELAXED, __HIP_MEMORY_SCOPE_AGENT);
    *(volatile unsigned*)(smem + MISC_LDS) = v;
  }
  __syncthreads();
  const unsigned call = __builtin_amdgcn_readfirstlane(*(volatile unsigned*)(smem + MISC_LDS));
  const size_t par_off = ABUF_OFF + (size_t)(call & 1u) * PAR_B;

  if (bid < NCOMM) {
    if (bid >= 7 * NWPP) return;
    const int q = 1 + bid % 7, w = bid / 7;
    const unsigned dst_rank = (unsigned)(p.rank + q) & (NR - 1);
    char* const dws = p.ws[dst_rank];
    char* const dbase = dws + par_off + (size_t)p.rank * SRC_B + (size_t)tid * 16;
    unsigned* const dflag = (unsigned*)(dws + FLAG_OFF + (size_t)p.rank * 256);
    const unsigned c = (unsigned)((tid & 7) ^ ((tid >> 4) & 7));
    const char* const sbase = p.x + (size_t)(tid >> 3) * XROW_B + c * 16;
    u32x4 v[8];
#pragma unroll
    for (int i = 0; i < 8; ++i) v[i] = *(const u32x4*)(sbase + (size_t)i * 64 * XROW_B + (size_t)w * 128);
    for (int s = w; s < NKS; s += NWPP) {
      char* d = dbase + (size_t)s * KS_B;
#pragma unroll
      for (int i = 0; i < 8; ++i)
        asm volatile("global_store_dwordx4 %0, %1, off sc0 sc1\n\ts_nop 1" :: "v"(d + i * 8192), "v"(v[i]) : "memory");
      const int sn = s + NWPP;
      if (sn < NKS) {
#pragma unroll
        for (int i = 0; i < 8; ++i) v[i] = *(const u32x4*)(sbase + (size_t)i * 64 * XROW_B + (size_t)sn * 128);
      }
      WAIT_V(0);
      __syncthreads();
      if (tid == 0)
        __hip_atomic_store(dflag + s, call, __ATOMIC_RELAXED, __HIP_MEMORY_SCOPE_SYSTEM);
    }
    STAMP(2);
    return;
  }

  const int xg = bid & 7;
  const int j = (bid - NCOMM) >> 3;
  const int cb = 2 * xg + (j & 1);
  const int ri = j >> 1;
  unsigned* const tmo = (unsigned*)(myws + TMO_OFF);
  const unsigned lds0 = (unsigned)(size_t)smem;

  unsigned boff[2];
#pragma unroll
  for (int i = 0; i < 2; ++i) {
    const unsigned k = (unsigned)(i * 32 + (tid >> 4));
    const unsigned h = (k & 3u) | (((k >> 3) & 1u) << 2);
    boff[i] = k * WROW_B + (((unsigned)(tid & 15) ^ (h << 1)) * 16u);
  }
  const char* const bbase = p.w + (size_t)cb * 256;

  if (j < 8) {
    const int lb = j >> 1;
    unsigned aoff[2];
    const unsigned c = (unsigned)((tid & 7) ^ ((tid >> 4) & 7));
#pragma unroll
    for (int i = 0; i < 2; ++i) aoff[i] = (unsigned)(i * 64 + (tid >> 3)) * XROW_B + c * 16u;
    run_unit<2>(smem, lds0, tid, lane, wid, call, tmo, p.x + (size_t)lb * 128 * XROW_B, 128u, aoff, bbase, boff,
                nullptr, p.out + ((size_t)(p.rank * MLOC + lb * 128) * NLOC + (size_t)cb * 128) * 2);
    STAMP(1);
    BAR();
  }
  {
    const int src = (p.rank + 1 + (ri >> 1)) & (NR - 1);
    const int rb = ri & 1;
    unsigned aoff[4];
#pragma unroll
    for (int i = 0; i < 4; ++i) aoff[i] = (unsigned)(i * 8192 + tid * 16);
    run_unit<4>(smem, lds0, tid, lane, wid, call, tmo, myws + par_off + (size_t)src * SRC_B + (size_t)rb * A_B,
                (unsigned)KS_B, aoff, bbase, boff, (const unsigned*)(myws + FLAG_OFF + (size_t)src * 256),
                p.out + ((size_t)(src * MLOC + rb * 256) * NLOC + (size_t)cb * 128) * 2);
  }
}

extern "C" void kernel_launch(int rank, int nranks, ncclComm_t comm,
                              hipStream_t stream,
                              void* const* d_in, const int* in_sizes, int n_in,
                              void* d_out, int out_size,
                              const int* in_dtypes, int out_dtype,
                              void* const* d_out_all,
                              void* const* d_ws_all, size_t ws_bytes) {
  (void)comm; (void)in_dtypes; (void)out_dtype; (void)d_out_all;
  if (nranks != NR || n_in != 2 || in_sizes[0] != MLOC * KTOT || in_sizes[1] != KTOT * NLOC ||
      out_size != NR * MLOC * NLOC || ws_bytes < WS_NEED || rank < 0 || rank >= NR) {
    fprintf(stderr, "ag_gemm_silu: unexpected problem shape\n");
    return;
  }
  static bool init[64] = {};
  int dev = 0;
  HIP_CHECK(hipGetDevice(&dev));
  if (dev < 0 || dev >= 64 || !init[dev]) {
    HIP_CHECK(hipFuncSetAttribute((const void*)ag_gemm_silu_kernel,
                                  hipFuncAttributeMaxDynamicSharedMemorySize, LDS_BYTES));
    if (dev >= 0 && dev < 64) init[dev] = true;
  }
  Params p;
  p.x = (const char*)d_in[0];
  p.w = (const char*)d_in[1];
  p.out = (char*)d_out;
  for (int i = 0; i < NR; ++i) p.ws[i] = (char*)d_ws_all[i];
  p.rank = rank;
  hipLaunchKernelGGL(ag_gemm_silu_kernel, dim3(NWG), dim3(512), LDS_BYTES, stream, p);
  HIP_CHECK(hipGetLastError());
}
```

```cpp
#include <hip/hip_runtime.h>
#include <rccl/rccl.h>
#include <cstdio>
#include <cstdint>

#define HIP_CHECK(expr) do { hipError_t _e = (expr); if (_e != hipSuccess) { \
    fprintf(stderr, "HIP error %s at %s:%d\n", hipGetErrorString(_e), __FILE__, __LINE__); \
    return; } } while (0)

typedef __attribute__((ext_vector_type(8))) short bf16x8;
typedef __attribute__((ext_vector_type(4))) short bf16x4;
typedef __attribute__((ext_vector_type(4))) float f32x4;
typedef __attribute__((ext_vector_type(2))) float f32x2;
typedef __attribute__((ext_vector_type(4))) unsigned u32x4;
typedef __attribute__((ext_vector_type(2))) unsigned u32x2;
typedef __attribute__((ext_vector_type(2))) __bf16 bf16x2_t;

namespace {

constexpr int NR = 8;
constexpr int KTOT = 4096, NLOC = 2048, MLOC = 512;
constexpr int BK = 64, NKS = KTOT / BK;
constexpr int NWG = 256, NCOMM = 32;
#define AG_NWPP 4
constexpr int NWPP = AG_NWPP;
static_assert(7 * NWPP <= NCOMM, "comm blocks");
constexpr int XROW_B = KTOT * 2;
constexpr int WROW_B = NLOC * 2;

constexpr int A_B = 32768, B_B = 16384, STAGE_B = A_B + B_B;
constexpr int FLG_LDS = 3 * STAGE_B;
constexpr int MISC_LDS = FLG_LDS + 2048;
constexpr int STRIP_B = 6144;
constexpr int STRIP_LDS = MISC_LDS + 64;
constexpr int LDS_BYTES = STRIP_LDS + 2 * STRIP_B;
static_assert(LDS_BYTES <= 163840, "LDS");

constexpr size_t CTR_OFF = 0;
constexpr size_t TMO_OFF = 32768;
constexpr size_t FLAG_OFF = 65536;
constexpr size_t ABUF_OFF = 1u << 20;
constexpr size_t SRC_B = 4u << 20, PAR_B = 32u << 20, KS_B = 65536;
constexpr size_t WS_NEED = ABUF_OFF + 2 * PAR_B;

constexpr unsigned long long DEADLINE_TICKS = 50000000ull;

struct Params {
  const char* x;
  const char* w;
  char* out;
  char* ws[NR];
  int rank;
};

#define WAIT_V(n) asm volatile("s_waitcnt vmcnt(" #n ")" ::: "memory")
#define WAIT_VN(n) asm volatile("s_waitcnt vmcnt(%0)" :: "n"(n) : "memory")
#define BAR() __builtin_amdgcn_s_barrier()
#define SCHED() __builtin_amdgcn_sched_barrier(0)
#define SETPRIO(x) __builtin_amdgcn_s_setprio(x)
#define DSR128(dst, addr, off) asm volatile("ds_read_b128 %0, %1 offset:" #off : "=v"(dst) : "v"(addr))
#define DSRTR(dst, addr, off) asm volatile("ds_read_b64_tr_b16 %0, %1 offset:" #off : "=v"(dst) : "v"(addr))

__device__ __forceinline__ float silu_f(float y) {
  float c = fminf(fmaxf(y, -60.f), 60.f);
  float e = __expf(-c);
  return y * __builtin_amdgcn_rcpf(1.f + e);
}

template <int NI>
__device__ __forceinline__ void run_unit(char* smem, const unsigned lds0, const int tid, const int lane, const int wid,
                                         const unsigned call, unsigned* tmo,
                                         const char* abase, const unsigned akstr, const unsigned* aoff,
                                         const char* bbase, const unsigned* boff, const unsigned* flg,
                                         char* otile, const char* sbase, const int sgroups, char* ostrip) {
  constexpr int NPA = (NI == 4) ? 4 : 2;
  constexpr int NP = NPA + 2;
  constexpr int NH = NP / 2;
  constexpr int NRD = 4 + 2 * NI;
  const int wm = (NI == 4) ? (wid >> 1) : (wid >> 2);
  const int wn = (NI == 4) ? (wid & 1) : (wid & 3);

  unsigned a_rd[2], b_rd[NI];
  {
    const unsigned r = (unsigned)(lane & 15), g = (unsigned)(lane >> 4), x = (unsigned)((lane >> 1) & 7);
    const unsigned base = (unsigned)(wm * 64 + r) * 128u;
    a_rd[0] = base + ((g ^ x) * 16u);
    a_rd[1] = base + (((4u + g) ^ x) * 16u);
    const unsigned q = (unsigned)((lane & 15) >> 2), pp = (unsigned)(lane & 3);
    const unsigned k = 8u * g + q;
    const unsigned h = q | ((g & 1u) << 2);
#pragma unroll
    for (int ni = 0; ni < NI; ++ni) {
      const unsigned cc = (unsigned)(wn * (2 * NI) + ni * 2) + (pp >> 1);
      b_rd[ni] = (unsigned)A_B + k * 256u + ((cc ^ (h << 1)) * 16u) + (pp & 1u) * 8u;
    }
  }

  const bool son = wm < sgroups;
  unsigned s_rd[2];
  {
    const unsigned r = (unsigned)(lane & 15), g = (unsigned)(lane >> 4), x = (unsigned)((lane >> 1) & 7);
    const unsigned base = (unsigned)(wm * 16 + r) * 128u;
    s_rd[0] = base + ((g ^ x) * 16u);
    s_rd[1] = base + (((4u + g) ^ x) * 16u);
  }
  unsigned soff;
  {
    const int row = tid >> 3;
    const int srow = row < sgroups * 16 ? row : sgroups * 16 - 1;
    soff = (unsigned)srow * XROW_B + (unsigned)((tid & 7) ^ ((tid >> 4) & 7)) * 16u;
  }
  auto strip_issue = [&](int s) {
    if (wid < 6)
      __builtin_amdgcn_global_load_lds((const unsigned*)(sbase + (size_t)s * 128 + soff),
                                       (unsigned*)(smem + STRIP_LDS + (s & 1) * STRIP_B + tid * 16), 16, 0, 0);
  };
  f32x4 accs[NI];
#pragma unroll
  for (int ni = 0; ni < NI; ++ni) accs[ni] = f32x4{0.f, 0.f, 0.f, 0.f};
  bf16x8 xs, ys;

  f32x4 acc[4][NI];
#pragma unroll
  for (int mi = 0; mi < 4; ++mi)
#pragma unroll
    for (int ni = 0; ni < NI; ++ni) acc[mi][ni] = f32x4{0.f, 0.f, 0.f, 0.f};

  int issued = 0;
  int ready = flg ? 0 : NKS;
  bool pend = false;
  int fill = -1;
  char* const myflg = smem + FLG_LDS + wid * 256 + lane * 4;
  const unsigned myflg_a = lds0 + (unsigned)(FLG_LDS + wid * 256 + lane * 4);

  auto gl = [&](int s, int i) {
    char* lb_ = smem + (s % 3) * STAGE_B + tid * 16;
    if (i < NPA) {
      const char* ab = abase + (size_t)s * akstr;
      __builtin_amdgcn_global_load_lds((const unsigned*)(ab + aoff[i]), (unsigned*)(lb_ + i * 8192), 16, 0, 0);
    } else {
      const char* bb = bbase + (size_t)s * (BK * WROW_B);
      __builtin_amdgcn_global_load_lds((const unsigned*)(bb + boff[i - NPA]), (unsigned*)(lb_ + A_B + (i - NPA) * 8192), 16, 0, 0);
    }
  };
  auto issue = [&](int s) {
#pragma unroll
    for (int i = 0; i < NP; ++i) gl(s, i);
  };
  auto poll_issue = [&]() {
    __builtin_amdgcn_global_load_lds(flg + lane, (unsigned*)myflg, 4, 0, 17);
  };
  auto poll_consume = [&]() {
    unsigned v;
    asm volatile("ds_read_b32 %0, %1\n\ts_waitcnt lgkmcnt(0)" : "=&v"(v) : "v"(myflg_a) : "memory");
    const unsigned long long m = __ballot(v >= call * (unsigned)NWPP);
    ready = (m == ~0ull) ? NKS : (int)__builtin_ctzll(~m);
  };

  bf16x8 xa[4], ya[4];
  bf16x4 xl[NI], xh[NI], yl[NI], yh[NI];
#define READ_ONE(A_, L_, H_, aa_, sbo_, koff_, k_) do {                                            \
    if ((k_) < 4) asm volatile("ds_read_b128 %0, %1 offset:%2" : "=v"(A_[(k_) & 3]) : "v"(aa_), "i"(((k_) & 3) * 2048)); \
    else if ((k_) < NRD) {                                                                          \
      const unsigned bb_ = (sbo_) + b_rd[((k_) - 4) >> 1];                                          \
      if (((k_) & 1) == 0) asm volatile("ds_read_b64_tr_b16 %0, %1 offset:%2" : "=v"(L_[((k_) - 4) >> 1]) : "v"(bb_), "i"(koff_)); \
      else asm volatile("ds_read_b64_tr_b16 %0, %1 offset:%2" : "=v"(H_[((k_) - 4) >> 1]) : "v"(bb_), "i"((koff_) + 1024)); } \
  } while (0)
#define READ_FRAGS(A_, L_, H_, st_, kk_, koff_) do {                                             \
    const unsigned sbo_ = lds0 + (unsigned)((st_) % 3) * (unsigned)STAGE_B;                         \
    const unsigned aa_ = sbo_ + a_rd[kk_];                                                          \
    _Pragma("unroll") for (int k = 0; k < NRD; ++k) READ_ONE(A_, L_, H_, aa_, sbo_, koff_, k);     \
  } while (0)
#define WAIT_FRAGS(n, A_, L_, H_) do {                                                            \
    if constexpr (NI == 4)                                                                          \
      asm volatile("s_waitcnt lgkmcnt(%12)" : "+v"(A_[0]), "+v"(A_[1]), "+v"(A_[2]), "+v"(A_[3]),  \
        "+v"(L_[0]), "+v"(L_[1]), "+v"(L_[2]), "+v"(L_[3]), "+v"(H_[0]), "+v"(H_[1]), "+v"(H_[2]), "+v"(H_[3]) : "n"(n)); \
    else                                                                                            \
      asm volatile("s_waitcnt lgkmcnt(%8)" : "+v"(A_[0]), "+v"(A_[1]), "+v"(A_[2]), "+v"(A_[3]),   \
        "+v"(L_[0]), "+v"(L_[1]), "+v"(H_[0]), "+v"(H_[1]) : "n"(n));                               \
  } while (0)
#define MMA_BLOCK(A_, L_, H_, S_, i0_, RA_, RL_, RH_, RS_, rst_, rkk_, rkoff_, rdo_) do {         \
    const unsigned rsbo_ = lds0 + (unsigned)((rst_) % 3) * (unsigned)STAGE_B;                       \
    const unsigned raa_ = rsbo_ + a_rd[rkk_];                                                       \
    SETPRIO(1);                                                                                     \
    _Pragma("unroll") for (int mi = 0; mi < 4; ++mi) {                                             \
      _Pragma("unroll") for (int ni = 0; ni < NI; ++ni) {                                          \
        const bf16x8 bb_ = __builtin_shufflevector(L_[ni], H_[ni], 0, 1, 2, 3, 4, 5, 6, 7);       \
        acc[mi][ni] = __builtin_amdgcn_mfma_f32_16x16x32_bf16(bb_, A_[mi], acc[mi][ni], 0, 0, 0);  \
        if (rdo_) READ_ONE(RA_, RL_, RH_, raa_, rsbo_, rkoff_, mi * NI + ni);                      \
        SCHED(); }                                                                                  \
      if (mi < NH) { if (fill >= 0) gl(fill, (i0_) + mi); SCHED(); } }                              \
    if (son) {                                                                                      \
      if (rdo_) asm volatile("ds_read_b128 %0, %1" : "=v"(RS_)                                     \
                             : "v"(lds0 + (unsigned)(STRIP_LDS + ((rst_) & 1) * STRIP_B) + s_rd[rkk_])); \
      _Pragma("unroll") for (int ni = 0; ni < NI; ++ni) {                                          \
        const bf16x8 bb_ = __builtin_shufflevector(L_[ni], H_[ni], 0, 1, 2, 3, 4, 5, 6, 7);       \
        accs[ni] = __builtin_amdgcn_mfma_f32_16x16x32_bf16(bb_, S_, accs[ni], 0, 0, 0); } }        \
    SETPRIO(0); } while (0)

  auto advance = [&](int t) {
    if (t + 1 < NKS) {
      if (issued <= t + 1) {
        WAIT_V(0);
        if (pend) { poll_consume(); pend = false; }
        if (ready <= issued) {
          const unsigned long long t0 = __builtin_amdgcn_s_memrealtime();
          for (;;) {
            poll_issue();
            WAIT_V(0);
            poll_consume();
            if (ready > issued) break;
            if (__hip_atomic_load(tmo, __ATOMIC_RELAXED, __HIP_MEMORY_SCOPE_AGENT) == call ||
                __builtin_amdgcn_s_memrealtime() - t0 > DEADLINE_TICKS) {
              __hip_atomic_store(tmo, call, __ATOMIC_RELAXED, __HIP_MEMORY_SCOPE_AGENT);
              ready = NKS;
              break;
            }
            __builtin_amdgcn_s_sleep(2);
          }
        }
        while (issued <= t + 2 && issued < ready) { issue(issued); ++issued; }
      }
      if (issued - 1 >= t + 2) { WAIT_VN(NP); } else { WAIT_V(0); }
      if (pend) { poll_consume(); pend = false; }
    }
    BAR();
    if (ready < NKS) { poll_issue(); pend = true; }
    if (t >= 0 && t + 2 < NKS) strip_issue(t + 2);
    if (t >= 0) {
      if (issued <= t + 3 && issued < ready) fill = issued;
    } else {
      while (issued <= t + 3 && issued < ready) { issue(issued); ++issued; }
    }
    if (t < 0) {
      READ_FRAGS(xa, xl, xh, 0, 0, 0);
      if (son) asm volatile("ds_read_b128 %0, %1" : "=v"(xs) : "v"(lds0 + (unsigned)STRIP_LDS + s_rd[0]));
    }
  };

  strip_issue(0);
  strip_issue(1);
  while (issued < 2 && issued < ready) { issue(issued); ++issued; }
  advance(-1);
  for (int t = 0; t < NKS; ++t) {
    WAIT_FRAGS(0, xa, xl, xh);
    asm volatile("" : "+v"(xs));
    MMA_BLOCK(xa, xl, xh, xs, NH, ya, yl, yh, ys, t, 1, 8192, true);
    if (fill >= 0) { ++issued; fill = -1; }
    while (issued <= t + 2 && issued < ready) { issue(issued); ++issued; }
    WAIT_FRAGS(0, ya, yl, yh);
    asm volatile("" : "+v"(ys));
    advance(t);
    MMA_BLOCK(ya, yl, yh, ys, 0, xa, xl, xh, xs, t + 1, 0, 0, t + 1 < NKS);
  }

  const bool bad = __hip_atomic_load(tmo, __ATOMIC_RELAXED, __HIP_MEMORY_SCOPE_AGENT) == call;
  char* obase = otile + ((size_t)(wm * 64 + (lane & 15)) * NLOC + (size_t)(wn * (16 * NI) + (lane >> 4) * 4)) * 2;
#pragma unroll
  for (int mi = 0; mi < 4; ++mi)
#pragma unroll
    for (int ni = 0; ni < NI; ++ni) {
      const f32x4 a = acc[mi][ni];
      const bf16x2_t lo = __builtin_convertvector(f32x2{silu_f(a[0]), silu_f(a[1])}, bf16x2_t);
      const bf16x2_t hi = __builtin_convertvector(f32x2{silu_f(a[2]), silu_f(a[3])}, bf16x2_t);
      u32x2 o;
      o[0] = __builtin_bit_cast(unsigned, lo);
      o[1] = __builtin_bit_cast(unsigned, hi);
      if (bad) { o[0] = 0x7FC07FC0u; o[1] = 0x7FC07FC0u; }
      *(u32x2*)(obase + (size_t)(mi * 16) * NLOC * 2 + ni * 32) = o;
    }
  if (son) {
    char* sb = ostrip + ((size_t)(wm * 16 + (lane & 15)) * NLOC + (size_t)(wn * (16 * NI) + (lane >> 4) * 4)) * 2;
#pragma unroll
    for (int ni = 0; ni < NI; ++ni) {
      const f32x4 a = accs[ni];
      const bf16x2_t lo = __builtin_convertvector(f32x2{silu_f(a[0]), silu_f(a[1])}, bf16x2_t);
      const bf16x2_t hi = __builtin_convertvector(f32x2{silu_f(a[2]), silu_f(a[3])}, bf16x2_t);
      u32x2 o;
      o[0] = __builtin_bit_cast(unsigned, lo);
      o[1] = __builtin_bit_cast(unsigned, hi);
      if (bad) { o[0] = 0x7FC07FC0u; o[1] = 0x7FC07FC0u; }
      *(u32x2*)(sb + ni * 32) = o;
    }
  }
#undef READ_FRAGS
#undef READ_ONE
#undef WAIT_FRAGS
#undef MMA_BLOCK
}

}

__global__ __launch_bounds__(512, 2) void ag_gemm_silu_kernel(Params p) {
  extern __shared__ __attribute__((aligned(16))) char smem[];
  const int tid = threadIdx.x;
  const int bid = blockIdx.x;
  const int lane = tid & 63;
  const int wid = __builtin_amdgcn_readfirstlane(tid >> 6);
  char* const myws = p.ws[p.rank];

#define STAMP(i) do { } while (0)
  STAMP(0);
  if (tid == 0) {
    unsigned* c = (unsigned*)(myws + CTR_OFF + (size_t)bid * 128);
    unsigned v = __hip_atomic_load(c, __ATOMIC_RELAXED, __HIP_MEMORY_SCOPE_AGENT) + 1u;
    __hip_atomic_store(c, v, __ATOMIC_RELAXED, __HIP_MEMORY_SCOPE_AGENT);
    *(volatile unsigned*)(smem + MISC_LDS) = v;
  }
  __syncthreads();
  const unsigned call = __builtin_amdgcn_readfirstlane(*(volatile unsigned*)(smem + MISC_LDS));
  const size_t par_off = ABUF_OFF + (size_t)(call & 1u) * PAR_B;

  if (bid < NCOMM) {
    if (bid >= 7 * NWPP) return;
    const int q = 1 + bid % 7, w = bid / 7;
    const unsigned dst_rank = (unsigned)(p.rank + q) & (NR - 1);
    char* const dws = p.ws[dst_rank];
    unsigned* const dflag = (unsigned*)(dws + FLAG_OFF + (size_t)p.rank * 256);
    constexpr int RW = MLOC / NWPP, NGR = RW * 8 / 512;
    static_assert(RW % 16 == 0 && NGR >= 1, "comm split");
    char* const dbase = dws + par_off + (size_t)p.rank * SRC_B + (size_t)(w * RW) * 128 + (size_t)tid * 16;
    const unsigned c = (unsigned)((tid & 7) ^ ((tid >> 4) & 7));
    const char* const sbase = p.x + (size_t)(w * RW + (tid >> 3)) * XROW_B + c * 16;
    u32x4 va[NGR], vb[NGR];
#define COMM_LD(V_, s_) do { _Pragma("unroll") for (int i = 0; i < NGR; ++i) V_[i] = *(const u32x4*)(sbase + (size_t)i * 64 * XROW_B + (size_t)(s_) * 128); } while (0)
#define COMM_ST(V_, s_) do { _Pragma("unroll") for (int i = 0; i < NGR; ++i)                       \
      asm volatile("global_store_dwordx4 %0, %1, off sc0 sc1\n\ts_nop 1" :: "v"(dbase + (size_t)(s_) * KS_B + i * 8192), "v"(V_[i]) : "memory"); } while (0)
    COMM_LD(va, 0);
    COMM_LD(vb, 1);
    COMM_ST(va, 0);
    for (int s = 0; s < NKS; s += 2) {
      if (s + 2 < NKS) COMM_LD(va, s + 2);
      COMM_ST(vb, s + 1);
      WAIT_VN(NGR);
      BAR();
      if (tid == 0) __hip_atomic_fetch_add(dflag + s, 1u, __ATOMIC_RELAXED, __HIP_MEMORY_SCOPE_SYSTEM);
      if (s + 2 < NKS) {
        if (s + 3 < NKS) COMM_LD(vb, s + 3);
        COMM_ST(va, s + 2);
        WAIT_VN(NGR);
      } else {
        WAIT_V(0);
      }
      BAR();
      if (tid == 0) __hip_atomic_fetch_add(dflag + s + 1, 1u, __ATOMIC_RELAXED, __HIP_MEMORY_SCOPE_SYSTEM);
    }
    STAMP(2);
    return;
  }

  const int xg = bid & 7;
  const int j = (bid - NCOMM) >> 3;
  const int cb = 2 * xg + (j & 1);
  const int ri = j >> 1;
  unsigned* const tmo = (unsigned*)(myws + TMO_OFF);
  const unsigned lds0 = (unsigned)(size_t)smem;

  unsigned boff[2];
#pragma unroll
  for (int i = 0; i < 2; ++i) {
    const unsigned k = (unsigned)(i * 32 + (tid >> 4));
    const unsigned h = (k & 3u) | (((k >> 3) & 1u) << 2);
    boff[i] = k * WROW_B + (((unsigned)(tid & 15) ^ (h << 1)) * 16u);
  }
  const char* const bbase = p.w + (size_t)cb * 256;

  {
    const int src = (p.rank + 1 + (ri >> 1)) & (NR - 1);
    const int rb = ri & 1;
    const int gs = ri < 4 ? 3 * ri : 12 + 2 * (ri - 4);
    unsigned aoff[4];
#pragma unroll
    for (int i = 0; i < 4; ++i) aoff[i] = (unsigned)(i * 8192 + tid * 16);
    run_unit<4>(smem, lds0, tid, lane, wid, call, tmo, myws + par_off + (size_t)src * SRC_B + (size_t)rb * A_B,
                (unsigned)KS_B, aoff, bbase, boff, (const unsigned*)(myws + FLAG_OFF + (size_t)src * 256),
                p.out + ((size_t)(src * MLOC + rb * 256) * NLOC + (size_t)cb * 128) * 2,
                p.x + (size_t)gs * 16 * XROW_B, ri < 4 ? 3 : 2,
                p.out + ((size_t)(p.rank * MLOC + gs * 16) * NLOC + (size_t)cb * 128) * 2);
  }
}

extern "C" void kernel_launch(int rank, int nranks, ncclComm_t comm,
                              hipStream_t stream,
                              void* const* d_in, const int* in_sizes, int n_in,
                              void* d_out, int out_size,
                              const int* in_dtypes, int out_dtype,
                              void* const* d_out_all,
                              void* const* d_ws_all, size_t ws_bytes) {
  (void)comm; (void)in_dtypes; (void)out_dtype; (void)d_out_all;
  if (nranks != NR || n_in != 2 || in_sizes[0] != MLOC * KTOT || in_sizes[1] != KTOT * NLOC ||
      out_size != NR * MLOC * NLOC || ws_bytes < WS_NEED || rank < 0 || rank >= NR) {
    fprintf(stderr, "ag_gemm_silu: unexpected problem shape\n");
    return;
  }
  static bool init[64] = {};
  int dev = 0;
  HIP_CHECK(hipGetDevice(&dev));
  if (dev < 0 || dev >= 64 || !init[dev]) {
    HIP_CHECK(hipFuncSetAttribute((const void*)ag_gemm_silu_kernel,
                                  hipFuncAttributeMaxDynamicSharedMemorySize, LDS_BYTES));
    if (dev >= 0 && dev < 64) init[dev] = true;
  }
  Params p;
  p.x = (const char*)d_in[0];
  p.w = (const char*)d_in[1];
  p.out = (char*)d_out;
  for (int i = 0; i < NR; ++i) p.ws[i] = (char*)d_ws_all[i];
  p.rank = rank;
  hipLaunchKernelGGL(ag_gemm_silu_kernel, dim3(NWG), dim3(512), LDS_BYTES, stream, p);
  HIP_CHECK(hipGetLastError());
}
```

```cpp
#define AG_EARLY 1
#include <hip/hip_runtime.h>
#include <rccl/rccl.h>
#include <cstdio>
#include <cstdint>

#define HIP_CHECK(expr) do { hipError_t _e = (expr); if (_e != hipSuccess) { \
    fprintf(stderr, "HIP error %s at %s:%d\n", hipGetErrorString(_e), __FILE__, __LINE__); \
    return; } } while (0)

typedef __attribute__((ext_vector_type(8))) short bf16x8;
typedef __attribute__((ext_vector_type(4))) short bf16x4;
typedef __attribute__((ext_vector_type(4))) float f32x4;
typedef __attribute__((ext_vector_type(2))) float f32x2;
typedef __attribute__((ext_vector_type(4))) unsigned u32x4;
typedef __attribute__((ext_vector_type(2))) unsigned u32x2;
typedef __attribute__((ext_vector_type(2))) __bf16 bf16x2_t;

namespace {

constexpr int NR = 8;
constexpr int KTOT = 4096, NLOC = 2048, MLOC = 512;
constexpr int BK = 64, NKS = KTOT / BK;
constexpr int NWG = 256, NCOMM = 32;
#define AG_NWPP 4
constexpr int NWPP = AG_NWPP;
static_assert(7 * NWPP <= NCOMM, "comm blocks");
constexpr int XROW_B = KTOT * 2;
constexpr int WROW_B = NLOC * 2;

constexpr int A_B = 32768, B_B = 16384, STAGE_B = A_B + B_B;
constexpr int FLG_LDS = 3 * STAGE_B;
constexpr int MISC_LDS = FLG_LDS + 2048;
constexpr int STRIP_B = 6144;
constexpr int STRIP_LDS = MISC_LDS + 64;
constexpr int LDS_BYTES = STRIP_LDS + 2 * STRIP_B;
static_assert(LDS_BYTES <= 163840, "LDS");

constexpr size_t CTR_OFF = 0;
constexpr size_t TMO_OFF = 32768;
constexpr size_t FLAG_OFF = 65536;
constexpr size_t ABUF_OFF = 1u << 20;
constexpr size_t SRC_B = 4u << 20, PAR_B = 32u << 20, KS_B = 65536;
constexpr size_t WS_NEED = ABUF_OFF + 2 * PAR_B;

constexpr unsigned long long DEADLINE_TICKS = 50000000ull;

struct Params {
  const char* x;
  const char* w;
  char* out;
  char* ws[NR];
  int rank;
};

#define WAIT_V(n) asm volatile("s_waitcnt vmcnt(" #n ")" ::: "memory")
#define WAIT_VN(n) asm volatile("s_waitcnt vmcnt(%0)" :: "n"(n) : "memory")
#define BAR() __builtin_amdgcn_s_barrier()
#define SCHED() __builtin_amdgcn_sched_barrier(0)
#define OSTORE(p_, v_) (*(p_) = (v_))
#define SETPRIO(x) __builtin_amdgcn_s_setprio(x)
#define DSR128(dst, addr, off) asm volatile("ds_read_b128 %0, %1 offset:" #off : "=v"(dst) : "v"(addr))
#define DSRTR(dst, addr, off) asm volatile("ds_read_b64_tr_b16 %0, %1 offset:" #off : "=v"(dst) : "v"(addr))

__device__ __forceinline__ float silu_f(float y) {
  float c = fminf(fmaxf(y, -60.f), 60.f);
  float e = __expf(-c);
  return y * __builtin_amdgcn_rcpf(1.f + e);
}

template <int NI>
__device__ __forceinline__ void run_unit(char* smem, const unsigned lds0, const int tid, const int lane, const int wid,
                                         const unsigned call, unsigned* tmo,
                                         const char* abase, const unsigned akstr, const unsigned* aoff,
                                         const char* bbase, const unsigned* boff, const unsigned* flg,
                                         char* otile, const char* sbase, const int sgroups, char* ostrip) {
  constexpr int NPA = (NI == 4) ? 4 : 2;
  constexpr int NP = NPA + 2;
  constexpr int NH = NP / 2;
  constexpr int NRD = 4 + 2 * NI;
  const int wm = (NI == 4) ? (wid >> 1) : (wid >> 2);
  const int wn = (NI == 4) ? (wid & 1) : (wid & 3);

  unsigned a_rd[2], b_rd[NI];
  {
    const unsigned r = (unsigned)(lane & 15), g = (unsigned)(lane >> 4), x = (unsigned)((lane >> 1) & 7);
    const unsigned base = (unsigned)(wm * 64 + r) * 128u;
    a_rd[0] = base + ((g ^ x) * 16u);
    a_rd[1] = base + (((4u + g) ^ x) * 16u);
    const unsigned q = (unsigned)((lane & 15) >> 2), pp = (unsigned)(lane & 3);
    const unsigned k = 8u * g + q;
    const unsigned h = q | ((g & 1u) << 2);
#pragma unroll
    for (int ni = 0; ni < NI; ++ni) {
      const unsigned cc = (unsigned)(wn * (2 * NI) + ni * 2) + (pp >> 1);
      b_rd[ni] = (unsigned)A_B + k * 256u + ((cc ^ (h << 1)) * 16u) + (pp & 1u) * 8u;
    }
  }

  const bool son = wm < sgroups;
  unsigned s_rd[2];
  {
    const unsigned r = (unsigned)(lane & 15), g = (unsigned)(lane >> 4), x = (unsigned)((lane >> 1) & 7);
    const unsigned base = (unsigned)(wm * 16 + r) * 128u;
    s_rd[0] = base + ((g ^ x) * 16u);
    s_rd[1] = base + (((4u + g) ^ x) * 16u);
  }
  unsigned soff;
  {
    const int row = tid >> 3;
    const int srow = row < sgroups * 16 ? row : sgroups * 16 - 1;
    soff = (unsigned)srow * XROW_B + (unsigned)((tid & 7) ^ ((tid >> 4) & 7)) * 16u;
  }
  auto strip_issue = [&](int s) {
    if (wid < 6)
      __builtin_amdgcn_global_load_lds((const unsigned*)(sbase + (size_t)s * 128 + soff),
                                       (unsigned*)(smem + STRIP_LDS + (s & 1) * STRIP_B + tid * 16), 16, 0, 0);
  };
  f32x4 accs[NI];
#pragma unroll
  for (int ni = 0; ni < NI; ++ni) accs[ni] = f32x4{0.f, 0.f, 0.f, 0.f};
  bf16x8 xs, ys;

  f32x4 acc[4][NI];
#pragma unroll
  for (int mi = 0; mi < 4; ++mi)
#pragma unroll
    for (int ni = 0; ni < NI; ++ni) acc[mi][ni] = f32x4{0.f, 0.f, 0.f, 0.f};

  int issued = 0;
  int ready = flg ? 0 : NKS;
  bool pend = false;
  int fill = -1;
  char* const myflg = smem + FLG_LDS + wid * 256 + lane * 4;
  const unsigned myflg_a = lds0 + (unsigned)(FLG_LDS + wid * 256 + lane * 4);

  auto gl = [&](int s, int i) {
    char* lb_ = smem + (s % 3) * STAGE_B + tid * 16;
    if (i < NPA) {
      const char* ab = abase + (size_t)s * akstr;
      __builtin_amdgcn_global_load_lds((const unsigned*)(ab + aoff[i]), (unsigned*)(lb_ + i * 8192), 16, 0, 0);
    } else {
      const char* bb = bbase + (size_t)s * (BK * WROW_B);
      __builtin_amdgcn_global_load_lds((const unsigned*)(bb + boff[i - NPA]), (unsigned*)(lb_ + A_B + (i - NPA) * 8192), 16, 0, 0);
    }
  };
  auto issue = [&](int s) {
#pragma unroll
    for (int i = 0; i < NP; ++i) gl(s, i);
  };
  auto poll_issue = [&]() {
    __builtin_amdgcn_global_load_lds(flg + lane, (unsigned*)myflg, 4, 0, 17);
  };
  auto poll_consume = [&]() {
    unsigned v;
    asm volatile("ds_read_b32 %0, %1\n\ts_waitcnt lgkmcnt(0)" : "=&v"(v) : "v"(myflg_a) : "memory");
    const unsigned long long m = __ballot(v >= call * (unsigned)NWPP);
    ready = (m == ~0ull) ? NKS : (int)__builtin_ctzll(~m);
  };

  bf16x8 xa[4], ya[4];
  bf16x4 xl[NI], xh[NI], yl[NI], yh[NI];
#define READ_ONE(A_, L_, H_, aa_, sbo_, koff_, k_) do {                                            \
    if ((k_) < 4) asm volatile("ds_read_b128 %0, %1 offset:%2" : "=v"(A_[(k_) & 3]) : "v"(aa_), "i"(((k_) & 3) * 2048)); \
    else if ((k_) < NRD) {                                                                          \
      const unsigned bb_ = (sbo_) + b_rd[((k_) - 4) >> 1];                                          \
      if (((k_) & 1) == 0) asm volatile("ds_read_b64_tr_b16 %0, %1 offset:%2" : "=v"(L_[((k_) - 4) >> 1]) : "v"(bb_), "i"(koff_)); \
      else asm volatile("ds_read_b64_tr_b16 %0, %1 offset:%2" : "=v"(H_[((k_) - 4) >> 1]) : "v"(bb_), "i"((koff_) + 1024)); } \
  } while (0)
#define READ_FRAGS(A_, L_, H_, st_, kk_, koff_) do {                                             \
    const unsigned sbo_ = lds0 + (unsigned)((st_) % 3) * (unsigned)STAGE_B;                         \
    const unsigned aa_ = sbo_ + a_rd[kk_];                                                          \
    _Pragma("unroll") for (int k = 0; k < NRD; ++k) READ_ONE(A_, L_, H_, aa_, sbo_, koff_, k);     \
  } while (0)
#define WAIT_FRAGS(n, A_, L_, H_) do {                                                            \
    if constexpr (NI == 4)                                                                          \
      asm volatile("s_waitcnt lgkmcnt(%12)" : "+v"(A_[0]), "+v"(A_[1]), "+v"(A_[2]), "+v"(A_[3]),  \
        "+v"(L_[0]), "+v"(L_[1]), "+v"(L_[2]), "+v"(L_[3]), "+v"(H_[0]), "+v"(H_[1]), "+v"(H_[2]), "+v"(H_[3]) : "n"(n)); \
    else                                                                                            \
      asm volatile("s_waitcnt lgkmcnt(%8)" : "+v"(A_[0]), "+v"(A_[1]), "+v"(A_[2]), "+v"(A_[3]),   \
        "+v"(L_[0]), "+v"(L_[1]), "+v"(H_[0]), "+v"(H_[1]) : "n"(n));                               \
  } while (0)
#define MMA_BLOCK(A_, L_, H_, S_, i0_, RA_, RL_, RH_, RS_, rst_, rkk_, rkoff_, rdo_) do {         \
    const unsigned rsbo_ = lds0 + (unsigned)((rst_) % 3) * (unsigned)STAGE_B;                       \
    const unsigned raa_ = rsbo_ + a_rd[rkk_];                                                       \
    SETPRIO(1);                                                                                     \
    _Pragma("unroll") for (int mi = 0; mi < 4; ++mi) {                                             \
      _Pragma("unroll") for (int ni = 0; ni < NI; ++ni) {                                          \
        const bf16x8 bb_ = __builtin_shufflevector(L_[ni], H_[ni], 0, 1, 2, 3, 4, 5, 6, 7);       \
        acc[mi][ni] = __builtin_amdgcn_mfma_f32_16x16x32_bf16(bb_, A_[mi], acc[mi][ni], 0, 0, 0);  \
        if (rdo_) READ_ONE(RA_, RL_, RH_, raa_, rsbo_, rkoff_, mi * NI + ni);                      \
        SCHED(); }                                                                                  \
      if (mi < NH) { if (fill >= 0) gl(fill, (i0_) + mi); SCHED(); } }                              \
    if (son) {                                                                                      \
      if (rdo_) asm volatile("ds_read_b128 %0, %1" : "=v"(RS_)                                     \
                             : "v"(lds0 + (unsigned)(STRIP_LDS + ((rst_) & 1) * STRIP_B) + s_rd[rkk_])); \
      _Pragma("unroll") for (int ni = 0; ni < NI; ++ni) {                                          \
        const bf16x8 bb_ = __builtin_shufflevector(L_[ni], H_[ni], 0, 1, 2, 3, 4, 5, 6, 7);       \
        accs[ni] = __builtin_amdgcn_mfma_f32_16x16x32_bf16(bb_, S_, accs[ni], 0, 0, 0); } }        \
    SETPRIO(0); } while (0)

  auto advance = [&](int t) {
    if (t + 1 < NKS) {
      if (issued <= t + 1) {
        WAIT_V(0);
        if (pend) { poll_consume(); pend = false; }
        if (ready <= issued) {
          const unsigned long long t0 = __builtin_amdgcn_s_memrealtime();
          for (;;) {
            poll_issue();
            WAIT_V(0);
            poll_consume();
            if (ready > issued) break;
            if (__hip_atomic_load(tmo, __ATOMIC_RELAXED, __HIP_MEMORY_SCOPE_AGENT) == call ||
                __builtin_amdgcn_s_memrealtime() - t0 > DEADLINE_TICKS) {
              __hip_atomic_store(tmo, call, __ATOMIC_RELAXED, __HIP_MEMORY_SCOPE_AGENT);
              ready = NKS;
              break;
            }
            __builtin_amdgcn_s_sleep(2);
          }
        }
        while (issued <= t + 2 && issued < ready) { issue(issued); ++issued; }
      }
      if (issued - 1 >= t + 2) { WAIT_VN(NP); } else { WAIT_V(0); }
      if (pend) { poll_consume(); pend = false; }
    }
    BAR();
    if (ready < NKS) { poll_issue(); pend = true; }
    if (t >= 0 && t + 2 < NKS) strip_issue(t + 2);
    if (t >= 0) {
      if (issued <= t + 3 && issued < ready) fill = issued;
    } else {
      while (issued <= t + 3 && issued < ready) { issue(issued); ++issued; }
    }
    if (t < 0) {
      READ_FRAGS(xa, xl, xh, 0, 0, 0);
      if (son) asm volatile("ds_read_b128 %0, %1" : "=v"(xs) : "v"(lds0 + (unsigned)STRIP_LDS + s_rd[0]));
    }
  };

  strip_issue(0);
  strip_issue(1);
  while (issued < 2 && issued < ready) { issue(issued); ++issued; }
  advance(-1);
  for (int t = 0; t < NKS; ++t) {
    WAIT_FRAGS(0, xa, xl, xh);
    asm volatile("" : "+v"(xs));
    MMA_BLOCK(xa, xl, xh, xs, NH, ya, yl, yh, ys, t, 1, 8192, true);
    if (fill >= 0) { ++issued; fill = -1; }
    while (issued <= t + 2 && issued < ready) { issue(issued); ++issued; }
    WAIT_FRAGS(0, ya, yl, yh);
    asm volatile("" : "+v"(ys));
    advance(t);
    MMA_BLOCK(ya, yl, yh, ys, 0, xa, xl, xh, xs, t + 1, 0, 0, t + 1 < NKS);
  }

  const bool bad = __hip_atomic_load(tmo, __ATOMIC_RELAXED, __HIP_MEMORY_SCOPE_AGENT) == call;
  char* obase = otile + ((size_t)(wm * 64 + (lane & 15)) * NLOC + (size_t)(wn * (16 * NI) + (lane >> 4) * 4)) * 2;
#pragma unroll
  for (int mi = 0; mi < 4; ++mi)
#pragma unroll
    for (int ni = 0; ni < NI; ++ni) {
      const f32x4 a = acc[mi][ni];
      const bf16x2_t lo = __builtin_convertvector(f32x2{silu_f(a[0]), silu_f(a[1])}, bf16x2_t);
      const bf16x2_t hi = __builtin_convertvector(f32x2{silu_f(a[2]), silu_f(a[3])}, bf16x2_t);
      u32x2 o;
      o[0] = __builtin_bit_cast(unsigned, lo);
      o[1] = __builtin_bit_cast(unsigned, hi);
      if (bad) { o[0] = 0x7FC07FC0u; o[1] = 0x7FC07FC0u; }
      OSTORE((u32x2*)(obase + (size_t)(mi * 16) * NLOC * 2 + ni * 32), o);
    }
  if (son) {
    char* sb = ostrip + ((size_t)(wm * 16 + (lane & 15)) * NLOC + (size_t)(wn * (16 * NI) + (lane >> 4) * 4)) * 2;
#pragma unroll
    for (int ni = 0; ni < NI; ++ni) {
      const f32x4 a = accs[ni];
      const bf16x2_t lo = __builtin_convertvector(f32x2{silu_f(a[0]), silu_f(a[1])}, bf16x2_t);
      const bf16x2_t hi = __builtin_convertvector(f32x2{silu_f(a[2]), silu_f(a[3])}, bf16x2_t);
      u32x2 o;
      o[0] = __builtin_bit_cast(unsigned, lo);
      o[1] = __builtin_bit_cast(unsigned, hi);
      if (bad) { o[0] = 0x7FC07FC0u; o[1] = 0x7FC07FC0u; }
      OSTORE((u32x2*)(sb + ni * 32), o);
    }
  }
#undef READ_FRAGS
#undef READ_ONE
#undef WAIT_FRAGS
#undef MMA_BLOCK
}

}

__global__ __launch_bounds__(512, 2) void ag_gemm_silu_kernel(Params p) {
  extern __shared__ __attribute__((aligned(16))) char smem[];
  const int tid = threadIdx.x;
  const int bid = blockIdx.x;
  const int lane = tid & 63;
  const int wid = __builtin_amdgcn_readfirstlane(tid >> 6);
  char* const myws = p.ws[p.rank];

#define STAMP(i) do { } while (0)
  STAMP(0);
  constexpr int RW0 = MLOC / NWPP, NGR0 = RW0 * 8 / 512;
  u32x4 va[NGR0], vb[NGR0];
  const char* sbase0 = nullptr;
  if (bid < 7 * NWPP) {
    const unsigned c0 = (unsigned)((tid & 7) ^ ((tid >> 4) & 7));
    sbase0 = p.x + (size_t)((bid / 7) * RW0 + (tid >> 3)) * XROW_B + c0 * 16;
#pragma unroll
    for (int i = 0; i < NGR0; ++i) { va[i] = *(const u32x4*)(sbase0 + (size_t)i * 64 * XROW_B); vb[i] = *(const u32x4*)(sbase0 + (size_t)i * 64 * XROW_B + 128); }
  }
  if (tid == 0) {
    unsigned* c = (unsigned*)(myws + CTR_OFF + (size_t)bid * 128);
    unsigned v = __hip_atomic_load(c, __ATOMIC_RELAXED, __HIP_MEMORY_SCOPE_AGENT) + 1u;
    __hip_atomic_store(c, v, __ATOMIC_RELAXED, __HIP_MEMORY_SCOPE_AGENT);
    *(volatile unsigned*)(smem + MISC_LDS) = v;
  }
  __syncthreads();
  const unsigned call = __builtin_amdgcn_readfirstlane(*(volatile unsigned*)(smem + MISC_LDS));
  const size_t par_off = ABUF_OFF + (size_t)(call & 1u) * PAR_B;

  if (bid < NCOMM) {
    if (bid >= 7 * NWPP) return;
    const int q = 1 + bid % 7, w = bid / 7;
    const unsigned dst_rank = (unsigned)(p.rank + q) & (NR - 1);
    char* const dws = p.ws[dst_rank];
    unsigned* const dflag = (unsigned*)(dws + FLAG_OFF + (size_t)p.rank * 256);
    constexpr int RW = MLOC / NWPP, NGR = RW * 8 / 512;
    static_assert(RW % 16 == 0 && NGR >= 1, "comm split");
    char* const dbase = dws + par_off + (size_t)p.rank * SRC_B + (size_t)(w * RW) * 128 + (size_t)tid * 16;
    const unsigned c = (unsigned)((tid & 7) ^ ((tid >> 4) & 7));
    const char* const sbase = p.x + (size_t)(w * RW + (tid >> 3)) * XROW_B + c * 16;
#define COMM_LD(V_, s_) do { _Pragma("unroll") for (int i = 0; i < NGR; ++i) V_[i] = *(const u32x4*)(sbase + (size_t)i * 64 * XROW_B + (size_t)(s_) * 128); } while (0)
#define COMM_ST(V_, s_) do { _Pragma("unroll") for (int i = 0; i < NGR; ++i)                       \
      asm volatile("global_store_dwordx4 %0, %1, off sc0 sc1\n\ts_nop 1" :: "v"(dbase + (size_t)(s_) * KS_B + i * 8192), "v"(V_[i]) : "memory"); } while (0)
    COMM_ST(va, 0);
    for (int s = 0; s < NKS; s += 2) {
      if (s + 2 < NKS) COMM_LD(va, s + 2);
      COMM_ST(vb, s + 1);
      WAIT_VN(NGR);
      BAR();
      if (tid == 0) __hip_atomic_fetch_add(dflag + s, 1u, __ATOMIC_RELAXED, __HIP_MEMORY_SCOPE_SYSTEM);
      if (s + 2 < NKS) {
        if (s + 3 < NKS) COMM_LD(vb, s + 3);
        COMM_ST(va, s + 2);
        WAIT_VN(NGR);
      } else {
        WAIT_V(0);
      }
      BAR();
      if (tid == 0) __hip_atomic_fetch_add(dflag + s + 1, 1u, __ATOMIC_RELAXED, __HIP_MEMORY_SCOPE_SYSTEM);
    }
    STAMP(2);
    return;
  }

  const int xg = bid & 7;
  const int j = (bid - NCOMM) >> 3;
  const int cb = 2 * xg + (j & 1);
  const int ri = j >> 1;
  unsigned* const tmo = (unsigned*)(myws + TMO_OFF);
  const unsigned lds0 = (unsigned)(size_t)smem;

  unsigned boff[2];
#pragma unroll
  for (int i = 0; i < 2; ++i) {
    const unsigned k = (unsigned)(i * 32 + (tid >> 4));
    const unsigned h = (k & 3u) | (((k >> 3) & 1u) << 2);
    boff[i] = k * WROW_B + (((unsigned)(tid & 15) ^ (h << 1)) * 16u);
  }
  const char* const bbase = p.w + (size_t)cb * 256;

  {
    const int src = (p.rank + 1 + (ri >> 1)) & (NR - 1);
    const int rb = ri & 1;
    const int gs = ri < 4 ? 3 * ri : 12 + 2 * (ri - 4);
    unsigned aoff[4];
#pragma unroll
    for (int i = 0; i < 4; ++i) aoff[i] = (unsigned)(i * 8192 + tid * 16);
    run_unit<4>(smem, lds0, tid, lane, wid, call, tmo, myws + par_off + (size_t)src * SRC_B + (size_t)rb * A_B,
                (unsigned)KS_B, aoff, bbase, boff, (const unsigned*)(myws + FLAG_OFF + (size_t)src * 256),
                p.out + ((size_t)(src * MLOC + rb * 256) * NLOC + (size_t)cb * 128) * 2,
                p.x + (size_t)gs * 16 * XROW_B, ri < 4 ? 3 : 2,
                p.out + ((size_t)(p.rank * MLOC + gs * 16) * NLOC + (size_t)cb * 128) * 2);
  }
}

extern "C" void kernel_launch(int rank, int nranks, ncclComm_t comm,
                              hipStream_t stream,
                              void* const* d_in, const int* in_sizes, int n_in,
                              void* d_out, int out_size,
                              const int* in_dtypes, int out_dtype,
                              void* const* d_out_all,
                              void* const* d_ws_all, size_t ws_bytes) {
  (void)comm; (void)in_dtypes; (void)out_dtype; (void)d_out_all;
  if (nranks != NR || n_in != 2 || in_sizes[0] != MLOC * KTOT || in_sizes[1] != KTOT * NLOC ||
      out_size != NR * MLOC * NLOC || ws_bytes < WS_NEED || rank < 0 || rank >= NR) {
    fprintf(stderr, "ag_gemm_silu: unexpected problem shape\n");
    return;
  }
  static bool init[64] = {};
  int dev = 0;
  HIP_CHECK(hipGetDevice(&dev));
  if (dev < 0 || dev >= 64 || !init[dev]) {
    HIP_CHECK(hipFuncSetAttribute((const void*)ag_gemm_silu_kernel,
                                  hipFuncAttributeMaxDynamicSharedMemorySize, LDS_BYTES));
    if (dev >= 0 && dev < 64) init[dev] = true;
  }
  Params p;
  p.x = (const char*)d_in[0];
  p.w = (const char*)d_in[1];
  p.out = (char*)d_out;
  for (int i = 0; i < NR; ++i) p.ws[i] = (char*)d_ws_all[i];
  p.rank = rank;
  hipLaunchKernelGGL(ag_gemm_silu_kernel, dim3(NWG), dim3(512), LDS_BYTES, stream, p);
  HIP_CHECK(hipGetLastError());
}
```
